```python
import numpy as np

kernel_name = "gpu_gemm_ar_m4096_k4096_n4096_e4m3e4m3_dqsilu"


ABSMAX_RTOL = 0.03

COMPUTE_DTYPE = "float32"
EPILOGUE = "dequant_silu"
IN_QUANT = None
OUT_QUANT = None
IN_FP8 = "e4m3e4m3"
IN_DTYPES = {"x": "e4m3", "w_mat": "e4m3",
             "scale_x": "float32", "scale_w": "float32"}
WORLD_SIZE = 8

def _snap_fp8(x, mant_bits, max_abs, min_normal):
    a = np.abs(x.astype(np.float64))
    _m, _e = np.frexp(a)
    q = np.where(a >= min_normal, np.ldexp(1.0, _e - 1 - mant_bits),
                 min_normal * 2.0 ** -mant_bits)
    snapped = np.minimum(np.round(a / q) * q, max_abs)
    return (np.sign(x) * snapped).astype(np.float32)

def setup_inputs():

    x = _snap_fp8(np.random.randn(4096, 4096).astype(np.float32), 3, 448.0, 2.0 ** -6)
    w_mat = _snap_fp8(np.random.randn(4096, 4096).astype(np.float32), 3, 448.0, 2.0 ** -6)
    scale_x = (0.02 * (1.0 + np.random.rand(1))).astype(np.float32)
    scale_w = (0.02 * (1.0 + np.random.rand(1))).astype(np.float32)
    return {"x": x, "w_mat": w_mat, "scale_x": scale_x, "scale_w": scale_w}

def reference(x, w_mat, scale_x, scale_w):
    acc = (x @ w_mat).astype(np.float32)
    y = (acc * (scale_x * scale_w)).astype(np.float32)
    return (y / (1.0 + np.exp(-np.clip(y, -60.0, 60.0)))).astype(np.float32)
```

```cpp
#include <hip/hip_runtime.h>
#include <rccl/rccl.h>
#include <cstdio>
#include <cstdint>

#define LAS __attribute__((address_space(3)))
typedef float v16f __attribute__((ext_vector_type(16)));
typedef float v4f __attribute__((ext_vector_type(4)));
typedef unsigned u32x4 __attribute__((ext_vector_type(4)));

#define POLL_SLEEP 2
#define SC " sc0 sc1"
#define OUT_AUX " sc0 sc1"

namespace {
constexpr int NR = 8, MM = 4096, NN = 4096, KSH = 512;
constexpr int NT = 32;
constexpr int NBLK = 32;
constexpr int NWG = 256;
constexpr int NTHREADS = 768;
constexpr size_t OFF_TICKET = 0;
constexpr size_t OFF_TMO = 32768;
constexpr size_t OFF_QUEUE = 36864;
constexpr size_t OFF_FLAGS = 65536;
constexpr size_t FLAG_T = 32, FLAG_ROW = NT * 32;
constexpr size_t OFF_STAGE = 1u << 20;
constexpr size_t STAGE_T = NBLK * 8192;
constexpr size_t STAGE_PAIR = NT * STAGE_T;
constexpr size_t STAGE_PARITY = 4 * STAGE_PAIR;
constexpr size_t WS_NEEDED = OFF_STAGE + 2 * STAGE_PARITY;
constexpr int SLOT_BYTES = 32768, NSLOT = 4;
constexpr int CTL = NSLOT * SLOT_BYTES;
constexpr int PSLOT = 4, PST = CTL + 256;
constexpr int LDS_BYTES = PST + PSLOT * 4096;
constexpr int C_FREE = 0  , C_FULL = 64  , C_CALL = 128, C_BAD = 132,
              C_PFULL = 136, C_PFREE = 140, C_PDESC = 144  ;
constexpr unsigned long long DEADLINE = 100000000ull;
static_assert(OFF_FLAGS + NBLK * FLAG_ROW <= OFF_STAGE, "flags");

struct Params {
  const unsigned char* x; const unsigned char* w; const float* sx; const float* sw; float* out;
  unsigned char* ws[NR]; int rank; int pad;
};

template <int N> __device__ __forceinline__ void wait_vm() { asm volatile("s_waitcnt vmcnt(%0)" ::"n"(N) : "memory"); }
__device__ __forceinline__ unsigned lds_ld(unsigned addr) { unsigned v; asm volatile("ds_read_b32 %0, %1\n\ts_waitcnt lgkmcnt(0)" : "=v"(v) : "v"(addr) : "memory"); return v; }
__device__ __forceinline__ int lds_ld_s(unsigned addr) { return __builtin_amdgcn_readfirstlane((int)lds_ld(addr)); }
__device__ __forceinline__ void lds_st(unsigned addr, unsigned v) { asm volatile("ds_write_b32 %0, %1" ::"v"(addr), "v"(v) : "memory"); }
__device__ __forceinline__ unsigned long long rtc() { return __builtin_amdgcn_s_memrealtime(); }

constexpr int NP = NT * NBLK;
constexpr unsigned TICKETS_PER_CALL = NP;

__device__ __forceinline__ void fetcher(const Params& P, int lane) {
  const int c = blockIdx.x;
  unsigned char* ws = P.ws[P.rank];
  const unsigned char* tp = ws + OFF_TICKET + (size_t)c * 128;
  const unsigned char* qp = ws + OFF_QUEUE + (size_t)(c >> 5) * 128;
  unsigned z = 0, one = 1;
  unsigned xo[4], wo[4];
#pragma unroll
  for (int q = 0; q < 4; ++q) { xo[q] = (64 * q + lane) * KSH; wo[q] = (4 * q + (lane >> 4)) * NN + (lane & 15) * 16; }
#define DRAW(R, A) asm volatile("s_mov_b64 exec, 1\n\tglobal_atomic_add " R ", %0, %1, %2 sc0\n\ts_mov_b64 exec, -1" ::"v"(z), "v"(one), "s"(A) : "memory", R)
  auto fetch = [&](unsigned u, int i) {
    const int t = u >> 5, b = (u >> 1) & 15; const bool isw = u & 1;
    const unsigned l = PST + (i & (PSLOT - 1)) * 4096;
    const unsigned char* g = isw ? P.w + (size_t)t * 16 * NN + b * 256 : P.x + (size_t)b * 256 * KSH + t * 16;
    const unsigned o0 = isw ? wo[0] : xo[0], o1 = isw ? wo[1] : xo[1], o2 = isw ? wo[2] : xo[2], o3 = isw ? wo[3] : xo[3];
    asm volatile("s_mov_b32 m0, %5\n\ts_nop 0\n\tglobal_load_lds_dwordx4 %0, %4\n\t"
                 "s_add_u32 m0, m0, 1024\n\ts_nop 0\n\tglobal_load_lds_dwordx4 %1, %4\n\t"
                 "s_add_u32 m0, m0, 1024\n\ts_nop 0\n\tglobal_load_lds_dwordx4 %2, %4\n\t"
                 "s_add_u32 m0, m0, 1024\n\ts_nop 0\n\tglobal_load_lds_dwordx4 %3, %4"
                 ::"v"(o0), "v"(o1), "v"(o2), "v"(o3), "s"(g), "s"(l) : "memory", "scc");
    lds_st(CTL + C_PDESC + (i & (PSLOT - 1)) * 4, u);
  };
  DRAW("v122", tp);
  fetch(c & 31, 0);
  unsigned call;
  asm volatile("s_waitcnt vmcnt(4)\n\tv_readfirstlane_b32 %0, v122" : "=s"(call) :: "memory");
  call += 1u;
  lds_st(CTL + C_CALL, call);
  DRAW("v121", qp);
  asm volatile("s_waitcnt lgkmcnt(0)" ::: "memory");
  __builtin_amdgcn_s_barrier();
  asm volatile("" ::: "memory");
  const unsigned base = (call - 1u) * TICKETS_PER_CALL - 32u;
  wait_vm<1>();
  lds_st(CTL + C_PFULL, 1);
  int i = 1;
  for (;; ++i) {
    unsigned tk;
    asm volatile("s_waitcnt vmcnt(0)\n\tv_readfirstlane_b32 %0, v121" : "=s"(tk) :: "memory");
    const unsigned u = tk - base;
    if (u >= (unsigned)NP) break;
    while (lds_ld_s(CTL + C_PFREE) < i) __builtin_amdgcn_s_sleep(1);
    fetch(u, i);
    DRAW("v121", qp);
    wait_vm<1>();
    lds_st(CTL + C_PFULL, i + 1);
  }
  lds_st(CTL + C_PDESC + (i & (PSLOT - 1)) * 4, 0xffffffffu);
  lds_st(CTL + C_PFULL, i + 1);
  asm volatile("s_waitcnt lgkmcnt(0)" ::: "memory");
}

template <int IMM> __device__ __forceinline__ void st16(unsigned voff, u32x4 d, const unsigned char* sbase) {
  asm volatile("global_store_dwordx4 %0, %1, %2 offset:%3" SC "\n\ts_nop 1" ::"v"(voff), "v"(d), "s"(sbase), "n"(IMM) : "memory");
}
__device__ __forceinline__ void flag_store(unsigned char* saddr, unsigned v) {
  unsigned z = 0;
  asm volatile("s_mov_b64 exec, 1\n\tglobal_store_dword %0, %1, %2" SC "\n\ts_mov_b64 exec, -1" ::"v"(z), "v"(v), "s"(saddr) : "memory");
}

__device__ __forceinline__ void storer(const Params& P, LAS unsigned char* lds, int lane, unsigned call) {
  const int dest = (P.rank + (blockIdx.x >> 5)) & 7;
  unsigned char* wsd = P.ws[dest];
  unsigned char* dst0 = wsd + OFF_STAGE + (size_t)(call & 1) * STAGE_PARITY + (size_t)(P.rank >> 1) * STAGE_PAIR + (P.rank & 1) * 512;
  unsigned char* flg0 = wsd + OFF_FLAGS + P.rank * 4;
  const unsigned so0 = (lane >> 5) * 1024 + (lane & 31) * 16, so1 = so0 + 4096;
  unsigned char* pending = nullptr;
  __builtin_amdgcn_s_setprio(2);
  for (int i = 0;; ++i) {
    if (lds_ld_s(CTL + C_PFULL) <= i) {
      if (pending) { wait_vm<0>(); flag_store(pending, call); pending = nullptr; }
      while (lds_ld_s(CTL + C_PFULL) <= i) __builtin_amdgcn_s_sleep(1);
    }
    const unsigned u = lds_ld_s(CTL + C_PDESC + (i & (PSLOT - 1)) * 4);
    if (u == 0xffffffffu) break;
    const int t = u >> 5, blk = (u & 1) * 16 + ((u >> 1) & 15);
    LAS unsigned char* sl = lds + PST + (i & (PSLOT - 1)) * 4096;
    u32x4 o[4];
    if (u & 1) {
      unsigned d[16];
#pragma unroll
      for (int k = 0; k < 16; ++k) d[k] = *(LAS const unsigned*)(sl + k * 256 + lane * 4);
#pragma unroll
      for (int g = 0; g < 4; ++g) {
        const unsigned d0 = d[4 * g], d1 = d[4 * g + 1], d2 = d[4 * g + 2], d3 = d[4 * g + 3];
        const unsigned t0 = __builtin_amdgcn_perm(d1, d0, 0x05010400u), t1 = __builtin_amdgcn_perm(d1, d0, 0x07030602u);
        const unsigned t2 = __builtin_amdgcn_perm(d3, d2, 0x05010400u), t3 = __builtin_amdgcn_perm(d3, d2, 0x07030602u);
        o[0][g] = __builtin_amdgcn_perm(t2, t0, 0x05040100u); o[1][g] = __builtin_amdgcn_perm(t2, t0, 0x07060302u);
        o[2][g] = __builtin_amdgcn_perm(t3, t1, 0x05040100u); o[3][g] = __builtin_amdgcn_perm(t3, t1, 0x07060302u);
      }
    } else {
#pragma unroll
      for (int q = 0; q < 4; ++q) o[q] = *(LAS const u32x4*)(sl + q * 1024 + lane * 16);
    }
    asm volatile("" : "+v"(o[0]), "+v"(o[1]), "+v"(o[2]), "+v"(o[3]) :: "memory");
    lds_st(CTL + C_PFREE, i + 1);
    unsigned char* d = dst0 + (size_t)t * STAGE_T + (size_t)blk * 8192;
    wait_vm<1>(); st16<0>(so0, o[0], d);
    wait_vm<1>();
    st16<2048>(so0, o[1], d);
    if (pending) { flag_store(pending, call); wait_vm<2>(); } else wait_vm<1>();
    st16<0>(so1, o[2], d);
    wait_vm<1>();
    st16<2048>(so1, o[3], d);
    pending = flg0 + (size_t)t * FLAG_T + (size_t)blk * FLAG_ROW;
  }
  wait_vm<0>();
  if (pending) flag_store(pending, call);
  wait_vm<0>();
}

__device__ __forceinline__ void loader(const Params& P, int part, int lane, unsigned call, int tm, int tn) {
  unsigned char* ws = P.ws[P.rank];
  const int blk = part ? 16 + tn : tm;
  const unsigned char* src = ws + OFF_STAGE + (size_t)(call & 1) * STAGE_PARITY + (size_t)blk * 8192;
  const unsigned char* flg = ws + OFF_FLAGS + (size_t)blk * FLAG_ROW;
  const unsigned char* tmo = ws + OFF_TMO;
  const unsigned vo0 = lane * 16, vo1 = lane * 16 + 4096;
  const unsigned fo = (lane >> 3) * (unsigned)FLAG_T + (lane & 7) * 4;
  const unsigned free_addr = CTL + C_FREE + (lane & 7) * 8, full_addr = CTL + C_FULL + (lane & 7) * 8 + part * 4;
  bool dead = false;
  __builtin_amdgcn_s_setprio(2);
#define GLDS8(G, L) asm volatile("s_mov_b32 m0, %3\n\ts_nop 0\n\t"                                                                            \
      "global_load_lds_dwordx4 %0, %2 offset:0" SC "\n\tglobal_load_lds_dwordx4 %0, %2 offset:1024" SC "\n\t"                                   \
      "global_load_lds_dwordx4 %0, %2 offset:2048" SC "\n\tglobal_load_lds_dwordx4 %0, %2 offset:3072" SC "\n\t"                                \
      "s_add_u32 m0, m0, 4096\n\ts_nop 0\n\t"                                                                                                  \
      "global_load_lds_dwordx4 %1, %2 offset:0" SC "\n\tglobal_load_lds_dwordx4 %1, %2 offset:1024" SC "\n\t"                                   \
      "global_load_lds_dwordx4 %1, %2 offset:2048" SC "\n\tglobal_load_lds_dwordx4 %1, %2 offset:3072" SC ::"v"(vo0), "v"(vo1), "s"(G), "s"(L) : "memory", "scc")
  int ready = 0, freed = NSLOT;
  for (int n = 0; n < 2 * NT; ++n) {
    const int t = n >> 1, h = n & 1;
    if (n >= ready) {
      const unsigned long long t0 = rtc();
      const unsigned fof = (t + (lane >> 3) < NT) ? fo : (unsigned)((lane & 7) * 4);
      for (;;) {
        unsigned v, tv;
        asm volatile("global_load_dword %0, %2, %3" SC "\n\tglobal_load_dword %1, %4, %5 sc1\n\ts_waitcnt vmcnt(0)"
                     : "=&v"(v), "=&v"(tv) : "v"(fof), "s"(flg + (size_t)t * FLAG_T), "v"(0u), "s"(tmo) : "memory");
        lds_st(full_addr, n);
        const unsigned long long nok = ~(__ballot(v >= call) >> (4 * h));
        const int lead = nok ? (__builtin_ctzll(nok) >> 2) : 16;
        if (!dead && (__builtin_amdgcn_readfirstlane(tv) == call || (lead == 0 && rtc() - t0 > DEADLINE))) {
          __hip_atomic_store((unsigned*)tmo, call, __ATOMIC_RELAXED, __HIP_MEMORY_SCOPE_AGENT); dead = true; lds_st(CTL + C_BAD, 1);
        }
        if (dead) { ready = 2 * NT; break; }
        if (lead > 0) { ready = n + lead; break; }
        __builtin_amdgcn_s_sleep(POLL_SLEEP);
      }
    }
    if (n >= freed) {
      for (;;) { const int f = (int)lds_ld(free_addr); if (__all(f >= n - (NSLOT - 1))) { freed = __all(f >= n - (NSLOT - 2)) ? n + 2 : n + 1; break; } __builtin_amdgcn_s_sleep(1); }
    }
    const unsigned lb = (n & (NSLOT - 1)) * SLOT_BYTES + part * 16384;
    const size_t go = (size_t)(2 * h) * STAGE_PAIR + (size_t)t * STAGE_T;
    GLDS8(src + go, lb); GLDS8(src + go + STAGE_PAIR, lb + 8192);
    wait_vm<16>();
    lds_st(full_addr, n);
  }
  wait_vm<0>();
  lds_st(full_addr, 2 * NT);
  asm volatile("s_waitcnt lgkmcnt(0)" ::: "memory");
}

#define RD(lo, hi, addr, off) "ds_read_b128 v[" lo "], " addr " offset:" off "\n\tds_read_b128 v[" hi "], " addr " offset:" off "+8192\n\t"
#define RD_P(o) RD("128:131", "132:135", "v160", o)
#define RD_Q(o) RD("136:139", "140:143", "v160", o)
#define RD_BX(o, i) RD("144:147", "148:151", "v161", o "+" #i "*2048")
#define RD_BY(o, i) RD("152:155", "156:159", "v161", o "+" #i "*2048")
#define MF(acc, a, b) "v_mfma_f32_32x32x64_f8f6f4 v[" acc "], v[" a "], v[" b "], v[" acc "]\n\t"
#define RP "128:135"
#define RQ "136:143"
#define RBX "144:151"
#define RBY "152:159"
#define LGKM(n) "s_waitcnt lgkmcnt(" #n ")\n\t"
#define CHECK_FULL "v_min_u32 v166, v166, v167\n\ts_nop 0\n\tv_readfirstlane_b32 %[tmp], v166\n\ts_cmp_gt_u32 %[tmp], %[n]\n\t"
#define READ_FULL "ds_read_b64 v[166:167], v164 offset:64\n\t"
#define SPIN(L) L ":\n\ts_sleep 1\n\t" READ_FULL LGKM(0) CHECK_FULL "s_cbranch_scc0 " L "\n\t"
#define TOGGLE "v_xor_b32 v160, 0x10000, v160\n\tv_xor_b32 v161, 0x10000, v161\n\t"
#define BODY2(L, cur, nxt, TG)                                                                                      \
  RD_BY(cur, 1)                                                                                                     \
  LGKM(4) MF("0:15", RP, RBX) LGKM(2) MF("64:79", RQ, RBX)                                                           \
  RD_BX(cur, 2)                                                                                                     \
  "s_add_u32 %[n], %[n], 1\n\tv_mov_b32 v165, %[n]\n\t"                                                             \
  LGKM(2) MF("16:31", RP, RBY) MF("80:95", RQ, RBY)                                                                  \
  RD_BY(cur, 3)                                                                                                     \
  "ds_write_b32 v164, v165\n\t" READ_FULL                                                                           \
  LGKM(4) MF("32:47", RP, RBX) MF("96:111", RQ, RBX)                                                                 \
  TG LGKM(0)                                                                                                        \
  "s_cmp_eq_u32 %[n], 64\n\ts_cbranch_scc1 Ldone%=\n\t"                                                             \
  CHECK_FULL "s_cbranch_scc0 Lslow" L "%=\n\t"                                                                      \
  RD_BX(nxt, 0) MF("48:63", RP, RBY) RD_P(nxt) MF("112:127", RQ, RBY) RD_Q(nxt "+4096")                              \
  "s_branch Lnext" L "%=\n"                                                                                         \
  "Lslow" L "%=:\n\t"                                                                                               \
  MF("48:63", RP, RBY) MF("112:127", RQ, RBY)                                                                        \
  SPIN("Lspin" L "%=")                                                                                              \
  RD_BX(nxt, 0) RD_P(nxt) RD_Q(nxt "+4096")                                                                         \
  "Lnext" L "%=:\n\t"
#define FRAG_CLOBBERS "v128", "v129", "v130", "v131", "v132", "v133", "v134", "v135", "v136", "v137", "v138", "v139", "v140", "v141", "v142", "v143", \
  "v144", "v145", "v146", "v147", "v148", "v149", "v150", "v151", "v152", "v153", "v154", "v155", "v156", "v157", "v158", "v159",                 \
  "v160", "v161", "v164", "v165", "v166", "v167"

__device__ __forceinline__ void epilogue(float* obase  , const v16f& c0, const v16f& c1, const v16f& c2, const v16f& c3, float sc, bool bad) {
  unsigned voff;
  asm volatile("v_mbcnt_lo_u32_b32 %0, -1, 0\n\tv_mbcnt_hi_u32_b32 %0, -1, %0" : "=v"(voff));
  voff = (voff >> 5) * (4 * NN * 4) + (voff & 31) * 16;
#define OST(Y, R) asm volatile("global_store_dwordx4 %0, %1, %2" OUT_AUX "\n\ts_nop 1" ::"v"(voff), "v"(Y), "s"(obase + (size_t)(((R) & 3) + 8 * ((R) >> 2)) * NN) : "memory")
#pragma unroll
  for (int r = 0; r < 16; ++r) {
    const float in[4] = {c0[r], c1[r], c2[r], c3[r]}; v4f y;
#pragma unroll
    for (int i = 0; i < 4; ++i) {
      const float v = in[i] * sc;
      const float cl = __builtin_amdgcn_fmed3f(v, -60.f, 60.f);
      const float e = __builtin_amdgcn_exp2f(cl * -1.44269504088896341f);
      y[i] = bad ? __builtin_nanf("") : v * __builtin_amdgcn_rcpf(1.0f + e);
    }
    OST(y, r);
  }
}
#define E8(OP) ".set e, 0\n\t.rept 8\n\t.set acc, (g>>3)*64 + (e&3)*16 + (g&7)*2 + (e>>2)\n\t.set tt, 128 + (g&1)*8 + e\n\t" OP "\n\t.set e, e+1\n\t.endr\n\t"
#define FAST_EPILOGUE                                                                                               \
  "v_mbcnt_lo_u32_b32 v146, -1, 0\n\tv_mbcnt_hi_u32_b32 v146, -1, v146\n\t"                                          \
  "v_lshrrev_b32 v147, 5, v146\n\tv_and_b32 v146, 31, v146\n\tv_lshlrev_b32 v146, 4, v146\n\tv_lshl_or_b32 v146, v147, 16, v146\n\t" \
  "v_mov_b32 v144, 0xc2ad1f97\n\tv_mov_b32 v145, 0x42ad1f97\n\t"                                                     \
  ".set g, 0\n\t.rept 16\n\t"                                                                                       \
  E8("v_mul_f32 v[tt], %[c1], v[acc]") E8("v_med3_f32 v[tt], v[tt], v144, v145") E8("v_exp_f32 v[tt], v[tt]")         \
  E8("v_fma_f32 v[tt], v[tt], %[inv], %[inv]") E8("v_rcp_f32 v[tt], v[tt]") E8("v_mul_f32 v[tt], v[acc], v[tt]")      \
  ".set r, (g&7)*2\n\t.set tt, 128 + (g&1)*8\n\t"                                                                   \
  "s_add_u32 vcc_lo, %[olo], ((g>>3)*128 + (r&3) + 8*(r>>2))*16384\n\ts_addc_u32 vcc_hi, %[ohi], 0\n\t"              \
  "global_store_dwordx4 v146, v[tt:tt+3], vcc" OUT_AUX "\n\t"                                                       \
  "s_add_u32 vcc_lo, vcc_lo, 16384\n\ts_addc_u32 vcc_hi, vcc_hi, 0\n\t"                                             \
  "global_store_dwordx4 v146, v[tt+4:tt+7], vcc" OUT_AUX "\n\t"                                                     \
  ".set g, g+1\n\t.endr\n\t"

__device__ __forceinline__ void compute(const Params& P, int wid, int tm, int tn) {
  const int wm = wid >> 1, wn = wid & 1;
  v16f c00, c01, c02, c03, c10, c11, c12, c13;
  unsigned n = 0, tmp;
  const unsigned sA = wm * 1024, sB = 16384 + wn * 1024, sC = CTL + wid * 8;
  int sc_bits = __builtin_amdgcn_readfirstlane(__builtin_bit_cast(int, P.sx[0] * P.sw[0]));
  asm volatile("" : "+s"(sc_bits));
  const float sc = __builtin_bit_cast(float, sc_bits);
  float* o = P.out + (size_t)(tm * 256 + wm * 32) * NN + tn * 256 + wn * 128;
  asm volatile(
      ".set i, 0\n\t.rept 128\n\tv_mov_b32 v[i], 0\n\t.set i, i+1\n\t.endr\n\t"
      "s_waitcnt lgkmcnt(0)\n\t"
      "v_mbcnt_lo_u32_b32 v160, -1, 0\n\tv_mbcnt_hi_u32_b32 v160, -1, v160\n\tv_lshlrev_b32 v160, 4, v160\n\t"
      "v_add_u32 v161, %[sb], v160\n\tv_add_u32 v160, %[sa], v160\n\tv_mov_b32 v164, %[sc]\n\t"
      READ_FULL LGKM(0) CHECK_FULL "s_cbranch_scc1 Lgo%=\n\t" SPIN("Lspin0%=") "Lgo%=:\n\t"
      RD_BX("0", 0) RD_P("0") RD_Q("0+4096")
      "Lloop%=:\n\t"
      BODY2("a", "0", "32768", "")
      BODY2("b", "32768", "0", TOGGLE)
      "s_branch Lloop%=\n"
      "Ldone%=:\n\t"
      MF("48:63", RP, RBY) MF("112:127", RQ, RBY)
      "s_nop 15\n\ts_nop 7\n\t"
      : [n] "+s"(n), [tmp] "=&s"(tmp), "={v[0:15]}"(c00), "={v[16:31]}"(c01), "={v[32:47]}"(c02), "={v[48:63]}"(c03),
        "={v[64:79]}"(c10), "={v[80:95]}"(c11), "={v[96:111]}"(c12), "={v[112:127]}"(c13)
      : [sa] "s"(sA), [sb] "s"(sB), [sc] "s"(sC)
      : "memory", "scc", FRAG_CLOBBERS);
  const bool bad = lds_ld_s(CTL + C_BAD) != 0;
  const float asc = __builtin_fabsf(sc);
  if (!bad && asc > 1e-30f && asc < 1e30f) {
    const int c1 = __builtin_amdgcn_readfirstlane(__builtin_bit_cast(int, sc * -1.44269504088896341f));
    const int inv = __builtin_amdgcn_readfirstlane(__builtin_bit_cast(int, 1.0f / sc));
    const unsigned olo = (unsigned)(uintptr_t)o, ohi = (unsigned)((uintptr_t)o >> 32);
    asm volatile(FAST_EPILOGUE
      : : "{v[0:15]}"(c00), "{v[16:31]}"(c01), "{v[32:47]}"(c02), "{v[48:63]}"(c03), "{v[64:79]}"(c10), "{v[80:95]}"(c11), "{v[96:111]}"(c12), "{v[112:127]}"(c13),
          [c1] "s"(c1), [inv] "s"(inv), [olo] "s"(olo), [ohi] "s"(ohi)
      : "memory", "scc", "vcc", FRAG_CLOBBERS);
  } else {
    epilogue(o, c00, c01, c02, c03, sc, bad);
    epilogue(o + (size_t)128 * NN, c10, c11, c12, c13, sc, bad);
  }
}

__global__ __launch_bounds__(NTHREADS, 3) void gemm_ag_kernel(Params P) {
  extern __shared__ __attribute__((aligned(16))) unsigned char smem[];
  LAS unsigned char* lds = (LAS unsigned char*)smem;
  const int wid = __builtin_amdgcn_readfirstlane(threadIdx.x >> 6), lane = threadIdx.x & 63;
  if (wid == 10) { fetcher(P, lane); return; }
  if (wid == 8) {
    if (lane < 32) lds_st(CTL + lane * 4, 0);
    if (lane < 3) lds_st(CTL + C_BAD + lane * 4, 0);
  }
  asm volatile("s_waitcnt lgkmcnt(0)" ::: "memory");
  __builtin_amdgcn_s_barrier();
  asm volatile("" ::: "memory");
  const int xg = blockIdx.x & 7, r = blockIdx.x >> 3;
  const int tm = (xg >> 1) * 4 + (r & 3), tn = (xg & 1) * 8 + (r >> 2);
  if (wid < 8) {
    compute(P, wid, tm, tn);
  } else {
    const unsigned call = __builtin_amdgcn_readfirstlane(lds_ld(CTL + C_CALL));
    if (wid == 11) storer(P, lds, lane, call);
    else loader(P, wid - 8, lane, call, tm, tn);
  }
}
}

extern "C" void kernel_launch(int rank, int nranks, ncclComm_t comm, hipStream_t stream,
                              void* const* d_in, const int* in_sizes, int n_in,
                              void* d_out, int out_size, const int* in_dtypes, int out_dtype,
                              void* const* d_out_all, void* const* d_ws_all, size_t ws_bytes) {
  (void)comm; (void)in_dtypes; (void)out_dtype; (void)d_out_all;
  if (nranks != NR || n_in != 4 || in_sizes[0] != MM * KSH || in_sizes[1] != KSH * NN || out_size != MM * NN || ws_bytes < WS_NEEDED || rank < 0 || rank >= NR) {
    fprintf(stderr, "kernel_launch: unexpected problem shape\n"); return;
  }
  static bool attr_set = false;
  if (!attr_set) {
    hipError_t e = hipFuncSetAttribute((const void*)gemm_ag_kernel, hipFuncAttributeMaxDynamicSharedMemorySize, LDS_BYTES);
    if (e != hipSuccess) { fprintf(stderr, "hipFuncSetAttribute: %s\n", hipGetErrorString(e)); return; }
    attr_set = true;
  }
  Params P{};
  P.x = (const unsigned char*)d_in[0]; P.w = (const unsigned char*)d_in[1]; P.sx = (const float*)d_in[2]; P.sw = (const float*)d_in[3];
  P.out = (float*)d_out; P.rank = rank; P.pad = 0;
  for (int p = 0; p < NR; ++p) P.ws[p] = (unsigned char*)d_ws_all[p];
  hipLaunchKernelGGL(gemm_ag_kernel, dim3(NWG), dim3(NTHREADS), LDS_BYTES, stream, P);
  hipError_t e = hipGetLastError();
  if (e != hipSuccess) fprintf(stderr, "launch: %s\n", hipGetErrorString(e));
}
```
